```python
import numpy as np

kernel_name = "gpu_gemm_rs_m2048_k2048_n2048_bf16_none"


ABSMAX_RTOL = 0.008

COMPUTE_DTYPE = "bfloat16"
EPILOGUE = "none"
IN_QUANT = None
OUT_QUANT = None
IN_DTYPES = {"x": "bfloat16", "w_mat": "bfloat16"}
WORLD_SIZE = 8

def _to_bf16(x):
    u = np.ascontiguousarray(x, dtype=np.float32).view(np.uint32)
    return ((u + 0x7FFF + ((u >> 16) & 1)) & 0xFFFF0000).view(np.float32)

def setup_inputs():

    x = _to_bf16(np.random.randn(2048, 2048).astype(np.float32))
    w_mat = _to_bf16((np.random.randn(2048, 2048) / np.sqrt(2048)).astype(np.float32))
    return {"x": x, "w_mat": w_mat}

def _epilogue(y):
    return y

def reference(x, w_mat):
    y = (x @ w_mat).astype(np.float32)
    return _epilogue(y)
```

```cpp
#include <hip/hip_runtime.h>
#include <rccl/rccl.h>
#include <cstdio>
#include <cstdint>

#define HIP_CHECK(expr) do { hipError_t _e = (expr); if (_e != hipSuccess) { \
    fprintf(stderr, "HIP error %s at %s:%d\n", hipGetErrorString(_e), __FILE__, __LINE__); \
    return; } } while (0)

typedef __attribute__((ext_vector_type(8))) short bf16x8;
typedef __attribute__((ext_vector_type(4))) float f32x4;
typedef __attribute__((ext_vector_type(4))) unsigned u32x4;
typedef __attribute__((ext_vector_type(2))) unsigned u32x2;

namespace {
constexpr int NR = 8;
constexpr int M_ = 2048, N_ = 2048, KL = 256;
constexpr int MS = M_ / NR;
constexpr int NWAVES = 512;
constexpr unsigned TILE_BYTES = 2048;
constexpr unsigned SLOT_BYTES = NWAVES * TILE_BYTES;
constexpr unsigned PAR_BYTES = NR * SLOT_BYTES;
constexpr unsigned CTR_OFF = 2 * PAR_BYTES;
constexpr unsigned WS_NEED = CTR_OFF + NWAVES * 64 + 4096;
constexpr unsigned long long DEADLINE_TICKS = 100000000ull;
constexpr int AUX_SYS = 17;

struct Params {
  void* ws[NR];
  const void* x;
  const void* w;
  void* out;
  int rank;
};

typedef __attribute__((ext_vector_type(2))) float f32x2;
typedef __attribute__((ext_vector_type(2))) __bf16 bf16x2;
__device__ __forceinline__ unsigned cvt_pk(float lo, float hi) {
  f32x2 f = {lo, hi};
  return __builtin_bit_cast(unsigned, __builtin_convertvector(f, bf16x2));
}
__device__ __forceinline__ unsigned enc(unsigned u) { return u == 0u ? 0x80008000u : u; }
__device__ __forceinline__ float bf_lo(unsigned u) { return __uint_as_float(u << 16); }
__device__ __forceinline__ float bf_hi(unsigned u) { return __uint_as_float(u & 0xffff0000u); }

__device__ __forceinline__ __amdgpu_buffer_rsrc_t mk_rsrc(const void* p, unsigned bytes) {
  return __builtin_amdgcn_make_buffer_rsrc(const_cast<void*>(p), 0, bytes, 0x00020000);
}
}

__global__ void __launch_bounds__(64, 2) gemm_rs_kernel(Params p) {
  const int lane = threadIdx.x;
  const int wv = blockIdx.x;
  const int xcd = wv & 7;
  const int idx = wv >> 3;
  const int col0 = (xcd * 4 + (idx & 3)) * 64;
  const int rb = idx >> 2;
  const int rank = p.rank;
  const int l15 = lane & 15, kq = lane >> 4;

  char* myws = (char*)p.ws[rank];
  unsigned* ctr = (unsigned*)(myws + CTR_OFF + wv * 64);
  const unsigned call = __builtin_amdgcn_readfirstlane(
      __hip_atomic_load(ctr, __ATOMIC_RELAXED, __HIP_MEMORY_SCOPE_SYSTEM)) + 1u;

  const __amdgpu_buffer_rsrc_t rx = mk_rsrc(p.x, M_ * KL * 2);
  const __amdgpu_buffer_rsrc_t rw = mk_rsrc(p.w, KL * N_ * 2);
  const __amdgpu_buffer_rsrc_t rme = mk_rsrc(myws, CTR_OFF);

  const unsigned a_off = (unsigned)((rb * 16 + l15) * KL + kq * 8) * 2u;
  auto load_a = [&](int d, u32x4 (&a)[8]) {
#pragma unroll
    for (int s = 0; s < 8; ++s)
      a[s] = __builtin_amdgcn_raw_buffer_load_b128(rx, a_off + s * 64, d * (MS * KL * 2), 0);
  };
  int q0 = xcd % 7;
  auto dest_of = [&](int t) { int q = q0 + t; q = q >= 7 ? q - 7 : q; return (rank + 1 + q) & 7; };

  u32x4 a_cur[8], a_nxt[8];
  load_a(dest_of(0), a_cur);

  u32x4 bfr[8][4];
  {
    const unsigned b_off = (unsigned)((kq * 8) * N_ + col0 + 4 * l15) * 2u;
#pragma unroll
    for (int s = 0; s < 8; ++s) {
      u32x2 raw[8];
#pragma unroll
      for (int i = 0; i < 8; ++i)
        raw[i] = __builtin_amdgcn_raw_buffer_load_b64(rw, b_off + (s * 32 + i) * (N_ * 2), 0, 0);
#pragma unroll
      for (int m = 0; m < 4; ++m) {
        bfr[s][0][m] = __builtin_amdgcn_perm(raw[2 * m + 1].x, raw[2 * m].x, 0x05040100u);
        bfr[s][1][m] = __builtin_amdgcn_perm(raw[2 * m + 1].x, raw[2 * m].x, 0x07060302u);
        bfr[s][2][m] = __builtin_amdgcn_perm(raw[2 * m + 1].y, raw[2 * m].y, 0x05040100u);
        bfr[s][3][m] = __builtin_amdgcn_perm(raw[2 * m + 1].y, raw[2 * m].y, 0x07060302u);
      }
    }
  }

  const unsigned par = call & 1u;
  const unsigned my_tile = wv * TILE_BYTES + lane * 16u;
  {
    const u32x4 z = {0u, 0u, 0u, 0u};
#pragma unroll
    for (int s = 0; s < NR; ++s) {
      __builtin_amdgcn_raw_buffer_store_b128(z, rme, my_tile, (par ^ 1u) * PAR_BYTES + s * SLOT_BYTES, AUX_SYS);
      __builtin_amdgcn_raw_buffer_store_b128(z, rme, my_tile + 1024u, (par ^ 1u) * PAR_BYTES + s * SLOT_BYTES, AUX_SYS);
    }
    if (lane == 0) __hip_atomic_store(ctr, call, __ATOMIC_RELAXED, __HIP_MEMORY_SCOPE_SYSTEM);
  }

  auto tile = [&](const u32x4 (&a)[8], f32x4 (&acc)[4]) {
#pragma unroll
    for (int j = 0; j < 4; ++j) acc[j] = f32x4{0.f, 0.f, 0.f, 0.f};
#pragma unroll
    for (int s = 0; s < 8; ++s)
#pragma unroll
      for (int j = 0; j < 4; ++j)
        acc[j] = __builtin_amdgcn_mfma_f32_16x16x32_bf16(
            __builtin_bit_cast(bf16x8, a[s]), __builtin_bit_cast(bf16x8, bfr[s][j]), acc[j], 0, 0, 0);
  };

  f32x4 acc[4];
#pragma unroll 1
  for (int t = 0; t < 7; ++t) {
    const int d = dest_of(t);
    const int dn = t < 6 ? dest_of(t + 1) : rank;
    load_a(dn, a_nxt);
    tile(a_cur, acc);
    u32x4 o0, o1;
    o0.x = enc(cvt_pk(acc[0][0], acc[1][0])); o0.y = enc(cvt_pk(acc[2][0], acc[3][0]));
    o0.z = enc(cvt_pk(acc[0][1], acc[1][1])); o0.w = enc(cvt_pk(acc[2][1], acc[3][1]));
    o1.x = enc(cvt_pk(acc[0][2], acc[1][2])); o1.y = enc(cvt_pk(acc[2][2], acc[3][2]));
    o1.z = enc(cvt_pk(acc[0][3], acc[1][3])); o1.w = enc(cvt_pk(acc[2][3], acc[3][3]));
    if (t == 0) {
      asm volatile("s_waitcnt vmcnt(8)" ::: "memory");
    }
    const __amdgpu_buffer_rsrc_t rd = mk_rsrc(p.ws[d], CTR_OFF);
    const unsigned so = par * PAR_BYTES + rank * SLOT_BYTES;
    __builtin_amdgcn_raw_buffer_store_b128(o0, rd, my_tile, so, AUX_SYS);
    __builtin_amdgcn_raw_buffer_store_b128(o1, rd, my_tile + 1024u, so, AUX_SYS);
#pragma unroll
    for (int s = 0; s < 8; ++s) a_cur[s] = a_nxt[s];
  }
  tile(a_cur, acc);

  u32x4 v[7][2];
  unsigned pending = 0x7fu;
  bool timed_out = false;
  const unsigned long long t0 = __builtin_amdgcn_s_memrealtime();
  while (true) {
#pragma unroll
    for (int t = 0; t < 7; ++t) {
      if (pending & (1u << t)) {
        int q = q0 + t; q = q >= 7 ? q - 7 : q;
        const int src = (rank + 7 - q) & 7;
        const unsigned so = par * PAR_BYTES + src * SLOT_BYTES;
        v[t][0] = __builtin_amdgcn_raw_buffer_load_b128(rme, my_tile, so, AUX_SYS);
        v[t][1] = __builtin_amdgcn_raw_buffer_load_b128(rme, my_tile + 1024u, so, AUX_SYS);
      }
    }
#pragma unroll
    for (int t = 0; t < 7; ++t) {
      if (pending & (1u << t)) {
        const bool ok = v[t][0].x && v[t][0].y && v[t][0].z && v[t][0].w &&
                        v[t][1].x && v[t][1].y && v[t][1].z && v[t][1].w;
        if (__all(ok)) pending &= ~(1u << t);
      }
    }
    if (!pending) break;
    if (__builtin_amdgcn_s_memrealtime() - t0 > DEADLINE_TICKS) { timed_out = true; break; }
    __builtin_amdgcn_s_sleep(4);
    asm volatile("" ::: "memory");
  }
#pragma unroll
  for (int t = 0; t < 7; ++t) {
    acc[0][0] += bf_lo(v[t][0].x); acc[1][0] += bf_hi(v[t][0].x);
    acc[2][0] += bf_lo(v[t][0].y); acc[3][0] += bf_hi(v[t][0].y);
    acc[0][1] += bf_lo(v[t][0].z); acc[1][1] += bf_hi(v[t][0].z);
    acc[2][1] += bf_lo(v[t][0].w); acc[3][1] += bf_hi(v[t][0].w);
    acc[0][2] += bf_lo(v[t][1].x); acc[1][2] += bf_hi(v[t][1].x);
    acc[2][2] += bf_lo(v[t][1].y); acc[3][2] += bf_hi(v[t][1].y);
    acc[0][3] += bf_lo(v[t][1].z); acc[1][3] += bf_hi(v[t][1].z);
    acc[2][3] += bf_lo(v[t][1].w); acc[3][3] += bf_hi(v[t][1].w);
  }
  const __amdgpu_buffer_rsrc_t ro = mk_rsrc(p.out, MS * N_ * 2);
#pragma unroll
  for (int r = 0; r < 4; ++r) {
    u32x2 o;
    o.x = cvt_pk(acc[0][r], acc[1][r]);
    o.y = cvt_pk(acc[2][r], acc[3][r]);
    if (timed_out) { o.x = 0x7fc07fc0u; o.y = 0x7fc07fc0u; }
    __builtin_amdgcn_raw_buffer_store_b64(o, ro, (unsigned)((rb * 16 + kq * 4 + r) * N_ + col0 + 4 * l15) * 2u, 0, 0);
  }
}

extern "C" void kernel_launch(int rank, int nranks, ncclComm_t comm,
                              hipStream_t stream,
                              void* const* d_in, const int* in_sizes, int n_in,
                              void* d_out, int out_size,
                              const int* in_dtypes, int out_dtype,
                              void* const* d_out_all,
                              void* const* d_ws_all, size_t ws_bytes) {
  if (nranks != NR || ws_bytes < WS_NEED || (unsigned)rank >= (unsigned)NR) {
    fprintf(stderr, "unexpected configuration\n");
    return;
  }
  Params p;
  for (int i = 0; i < NR; ++i) p.ws[i] = d_ws_all[i];
  p.x = d_in[0];
  p.w = d_in[1];
  p.out = d_out;
  p.rank = rank;
  gemm_rs_kernel<<<NWAVES, 64, 0, stream>>>(p);
  HIP_CHECK(hipGetLastError());
}
```

```cpp
#include <hip/hip_runtime.h>
#include <rccl/rccl.h>
#include <cstdio>
#include <cstdint>

#define HIP_CHECK(expr) do { hipError_t _e = (expr); if (_e != hipSuccess) { \
    fprintf(stderr, "HIP error %s at %s:%d\n", hipGetErrorString(_e), __FILE__, __LINE__); \
    return; } } while (0)

typedef __attribute__((ext_vector_type(8))) short bf16x8;
typedef __attribute__((ext_vector_type(4))) float f32x4;
typedef __attribute__((ext_vector_type(4))) unsigned u32x4;
typedef __attribute__((ext_vector_type(2))) unsigned u32x2;
typedef __attribute__((ext_vector_type(2))) float f32x2;
typedef __attribute__((ext_vector_type(2))) __bf16 bf16x2;

namespace {
constexpr int NR = 8;
constexpr int M_ = 2048, N_ = 2048, KL = 256;
constexpr int MS = M_ / NR;
constexpr int NWAVES = 512;
constexpr unsigned TILE_BYTES = 2048;
constexpr unsigned SLOT_BYTES = NWAVES * TILE_BYTES;
constexpr unsigned PAR_BYTES = NR * SLOT_BYTES;
constexpr unsigned CTR_OFF = 2 * PAR_BYTES;
constexpr unsigned WS_NEED = CTR_OFF + NWAVES * 64 + 4096;
constexpr unsigned long long DEADLINE_TICKS = 100000000ull;
constexpr int AUX_SYS = 17;

struct Params {
  void* ws[NR];
  const void* x;
  const void* w;
  void* out;
  int rank;
};

__device__ __forceinline__ unsigned cvt_pk(float lo, float hi) {
  f32x2 f = {lo, hi};
  return __builtin_bit_cast(unsigned, __builtin_convertvector(f, bf16x2));
}
__device__ __forceinline__ unsigned enc(unsigned u) { return u == 0u ? 0x80008000u : u; }
__device__ __forceinline__ float bf_lo(unsigned u) { return __uint_as_float(u << 16); }
__device__ __forceinline__ float bf_hi(unsigned u) { return __uint_as_float(u & 0xffff0000u); }

__device__ __forceinline__ __amdgpu_buffer_rsrc_t mk_rsrc(const void* p, unsigned bytes) {
  return __builtin_amdgcn_make_buffer_rsrc(const_cast<void*>(p), 0, bytes, 0x00020000);
}
}

__global__ void __launch_bounds__(64, 2) gemm_rs_kernel(Params p) {
  const int lane = threadIdx.x;
  const int wv = blockIdx.x;
  const int xcd = wv & 7;
  const int idx = wv >> 3;
  const int col0 = (xcd * 4 + (idx & 3)) * 64;
  const int rb = idx >> 2;
  const int rank = p.rank;
  const int l15 = lane & 15, kq = lane >> 4;

  char* myws = (char*)p.ws[rank];
  unsigned* ctr = (unsigned*)(myws + CTR_OFF + wv * 64);
  const unsigned call_v = __hip_atomic_load(ctr, __ATOMIC_RELAXED, __HIP_MEMORY_SCOPE_SYSTEM);

  const __amdgpu_buffer_rsrc_t rx = mk_rsrc(p.x, M_ * KL * 2);
  const __amdgpu_buffer_rsrc_t rw = mk_rsrc(p.w, KL * N_ * 2);
  const __amdgpu_buffer_rsrc_t rme = mk_rsrc(myws, CTR_OFF);

  const int q0 = wv % 7;
  auto dest_of = [&](int t) { int q = q0 + t; q = q >= 7 ? q - 7 : q; return t == 7 ? rank : ((rank + 1 + q) & 7); };

  const unsigned a_off = (unsigned)((rb * 16 + l15) * KL + kq * 8) * 2u;
  u32x4 a0[8];
#pragma unroll
  for (int s = 0; s < 8; ++s)
    a0[s] = __builtin_amdgcn_raw_buffer_load_b128(rx, a_off + s * 64, dest_of(0) * (MS * KL * 2), 0);

  u32x2 raw[8][8];
  {
    const unsigned b_off = (unsigned)((kq * 8) * N_ + col0 + 4 * l15) * 2u;
#pragma unroll
    for (int s = 0; s < 8; ++s)
#pragma unroll
      for (int i = 0; i < 8; ++i)
        raw[s][i] = __builtin_amdgcn_raw_buffer_load_b64(rw, b_off + (s * 32 + i) * (N_ * 2), 0, 0);
  }
  u32x4 a1[8], a2[8];
  auto load_a = [&](int t, u32x4 (&a)[8]) {
#pragma unroll
    for (int s = 0; s < 8; ++s)
      a[s] = __builtin_amdgcn_raw_buffer_load_b128(rx, a_off + s * 64, dest_of(t) * (MS * KL * 2), 0);
  };
  load_a(1, a1);
  __builtin_amdgcn_sched_barrier(0);
  u32x4 bfr[8][4];
#pragma unroll
  for (int s = 0; s < 8; ++s)
#pragma unroll
    for (int m = 0; m < 4; ++m) {
      bfr[s][0][m] = __builtin_amdgcn_perm(raw[s][2 * m + 1].x, raw[s][2 * m].x, 0x05040100u);
      bfr[s][1][m] = __builtin_amdgcn_perm(raw[s][2 * m + 1].x, raw[s][2 * m].x, 0x07060302u);
      bfr[s][2][m] = __builtin_amdgcn_perm(raw[s][2 * m + 1].y, raw[s][2 * m].y, 0x05040100u);
      bfr[s][3][m] = __builtin_amdgcn_perm(raw[s][2 * m + 1].y, raw[s][2 * m].y, 0x07060302u);
    }

  const unsigned call = __builtin_amdgcn_readfirstlane(call_v) + 1u;
  const unsigned par = call & 1u;
  const unsigned my_tile = wv * TILE_BYTES + lane * 16u;

  auto tile = [&](const u32x4 (&a)[8], f32x4 (&acc)[4]) {
#pragma unroll
    for (int j = 0; j < 4; ++j) acc[j] = f32x4{0.f, 0.f, 0.f, 0.f};
#pragma unroll
    for (int s = 0; s < 8; ++s)
#pragma unroll
      for (int j = 0; j < 4; ++j)
        acc[j] = __builtin_amdgcn_mfma_f32_16x16x32_bf16(
            __builtin_bit_cast(bf16x8, a[s]), __builtin_bit_cast(bf16x8, bfr[s][j]), acc[j], 0, 0, 0);
  };
  auto push = [&](int d, const f32x4 (&acc)[4]) {
    u32x4 o0, o1;
    o0.x = enc(cvt_pk(acc[0][0], acc[1][0])); o0.y = enc(cvt_pk(acc[2][0], acc[3][0]));
    o0.z = enc(cvt_pk(acc[0][1], acc[1][1])); o0.w = enc(cvt_pk(acc[2][1], acc[3][1]));
    o1.x = enc(cvt_pk(acc[0][2], acc[1][2])); o1.y = enc(cvt_pk(acc[2][2], acc[3][2]));
    o1.z = enc(cvt_pk(acc[0][3], acc[1][3])); o1.w = enc(cvt_pk(acc[2][3], acc[3][3]));
    const __amdgpu_buffer_rsrc_t rd = mk_rsrc(p.ws[d], CTR_OFF);
    const unsigned so = par * PAR_BYTES + rank * SLOT_BYTES;
    __builtin_amdgcn_raw_buffer_store_b128(o0, rd, my_tile, so, AUX_SYS);
    __builtin_amdgcn_raw_buffer_store_b128(o1, rd, my_tile + 1024u, so, AUX_SYS);
  };

  f32x4 acc[4];
#define SB __builtin_amdgcn_sched_barrier(0)
  SB;
  tile(a0, acc); SB; load_a(2, a2); SB; push(dest_of(0), acc); SB;
  tile(a1, acc); SB; load_a(3, a0); SB; push(dest_of(1), acc); SB;
  tile(a2, acc); SB; load_a(4, a1); SB; push(dest_of(2), acc); SB;
  tile(a0, acc); SB; load_a(5, a2); SB; push(dest_of(3), acc); SB;
  tile(a1, acc); SB; load_a(6, a0); SB; push(dest_of(4), acc); SB;
  tile(a2, acc); SB; load_a(7, a1); SB; push(dest_of(5), acc); SB;
  tile(a0, acc); SB; push(dest_of(6), acc); SB;
  tile(a1, acc);
  if (lane == 0) __hip_atomic_store(ctr, call, __ATOMIC_RELAXED, __HIP_MEMORY_SCOPE_SYSTEM);

  u32x4 v[7][2];
  unsigned pending = 0x7fu;
  bool timed_out = false;
  const unsigned long long t0 = __builtin_amdgcn_s_memrealtime();
  while (true) {
#pragma unroll
    for (int t = 0; t < 7; ++t) {
      if (pending & (1u << t)) {
        int q = q0 + t; q = q >= 7 ? q - 7 : q;
        const int src = (rank + 7 - q) & 7;
        const unsigned so = par * PAR_BYTES + src * SLOT_BYTES;
        v[t][0] = __builtin_amdgcn_raw_buffer_load_b128(rme, my_tile, so, AUX_SYS);
        v[t][1] = __builtin_amdgcn_raw_buffer_load_b128(rme, my_tile + 1024u, so, AUX_SYS);
      }
    }
#pragma unroll
    for (int t = 0; t < 7; ++t) {
      if (pending & (1u << t)) {
        const bool ok = v[t][0].x && v[t][0].y && v[t][0].z && v[t][0].w &&
                        v[t][1].x && v[t][1].y && v[t][1].z && v[t][1].w;
        if (__all(ok)) pending &= ~(1u << t);
      }
    }
    if (!pending) break;
    if (__builtin_amdgcn_s_memrealtime() - t0 > DEADLINE_TICKS) { timed_out = true; break; }
    __builtin_amdgcn_s_sleep(4);
    asm volatile("" ::: "memory");
  }
  {
    const u32x4 z = {0u, 0u, 0u, 0u};
#pragma unroll
    for (int t = 0; t < 7; ++t) {
      int q = q0 + t; q = q >= 7 ? q - 7 : q;
      const int src = (rank + 7 - q) & 7;
      const unsigned so = par * PAR_BYTES + src * SLOT_BYTES;
      __builtin_amdgcn_raw_buffer_store_b128(z, rme, my_tile, so, AUX_SYS);
      __builtin_amdgcn_raw_buffer_store_b128(z, rme, my_tile + 1024u, so, AUX_SYS);
    }
  }
#pragma unroll
  for (int t = 0; t < 7; ++t) {
    acc[0][0] += bf_lo(v[t][0].x); acc[1][0] += bf_hi(v[t][0].x);
    acc[2][0] += bf_lo(v[t][0].y); acc[3][0] += bf_hi(v[t][0].y);
    acc[0][1] += bf_lo(v[t][0].z); acc[1][1] += bf_hi(v[t][0].z);
    acc[2][1] += bf_lo(v[t][0].w); acc[3][1] += bf_hi(v[t][0].w);
    acc[0][2] += bf_lo(v[t][1].x); acc[1][2] += bf_hi(v[t][1].x);
    acc[2][2] += bf_lo(v[t][1].y); acc[3][2] += bf_hi(v[t][1].y);
    acc[0][3] += bf_lo(v[t][1].z); acc[1][3] += bf_hi(v[t][1].z);
    acc[2][3] += bf_lo(v[t][1].w); acc[3][3] += bf_hi(v[t][1].w);
  }
  const __amdgpu_buffer_rsrc_t ro = mk_rsrc(p.out, MS * N_ * 2);
#pragma unroll
  for (int r = 0; r < 4; ++r) {
    u32x2 o;
    o.x = cvt_pk(acc[0][r], acc[1][r]);
    o.y = cvt_pk(acc[2][r], acc[3][r]);
    if (timed_out) { o.x = 0x7fc07fc0u; o.y = 0x7fc07fc0u; }
    __builtin_amdgcn_raw_buffer_store_b64(o, ro, (unsigned)((rb * 16 + kq * 4 + r) * N_ + col0 + 4 * l15) * 2u, 0, 0);
  }
}

extern "C" void kernel_launch(int rank, int nranks, ncclComm_t comm,
                              hipStream_t stream,
                              void* const* d_in, const int* in_sizes, int n_in,
                              void* d_out, int out_size,
                              const int* in_dtypes, int out_dtype,
                              void* const* d_out_all,
                              void* const* d_ws_all, size_t ws_bytes) {
  if (nranks != NR || ws_bytes < WS_NEED || (unsigned)rank >= (unsigned)NR) {
    fprintf(stderr, "unexpected configuration\n");
    return;
  }
  Params p;
  for (int i = 0; i < NR; ++i) p.ws[i] = d_ws_all[i];
  p.x = d_in[0];
  p.w = d_in[1];
  p.out = d_out;
  p.rank = rank;
  gemm_rs_kernel<<<NWAVES, 64, 0, stream>>>(p);
  HIP_CHECK(hipGetLastError());
}
```

```cpp
#include <hip/hip_runtime.h>
#include <rccl/rccl.h>
#include <cstdio>
#include <cstdint>

#define HIP_CHECK(expr) do { hipError_t _e = (expr); if (_e != hipSuccess) { \
    fprintf(stderr, "HIP error %s at %s:%d\n", hipGetErrorString(_e), __FILE__, __LINE__); \
    return; } } while (0)

typedef __attribute__((ext_vector_type(8))) short bf16x8;
typedef __attribute__((ext_vector_type(4))) float f32x4;
typedef __attribute__((ext_vector_type(4))) unsigned u32x4;
typedef __attribute__((ext_vector_type(2))) unsigned u32x2;
typedef __attribute__((ext_vector_type(2))) float f32x2;
typedef __attribute__((ext_vector_type(2))) __bf16 bf16x2;

namespace {
constexpr int NR = 8;
constexpr int M_ = 2048, N_ = 2048, KL = 256;
constexpr int MS = M_ / NR;
constexpr int NWAVES = 512;
constexpr unsigned TILE_BYTES = 2048;
constexpr unsigned SLOT_BYTES = NWAVES * TILE_BYTES;
constexpr unsigned PAR_BYTES = NR * SLOT_BYTES;
constexpr unsigned CTR_OFF = 2 * PAR_BYTES;
constexpr unsigned WS_NEED = CTR_OFF + NWAVES * 64 + 4096;
constexpr unsigned long long DEADLINE_TICKS = 100000000ull;
constexpr int AUX_SYS = 17;

struct Params {
  void* ws[NR];
  const void* x;
  const void* w;
  void* out;
  int rank;
};

__device__ __forceinline__ unsigned cvt_pk(float lo, float hi) {
  f32x2 f = {lo, hi};
  return __builtin_bit_cast(unsigned, __builtin_convertvector(f, bf16x2));
}
__device__ __forceinline__ unsigned enc(unsigned u) { return u == 0u ? 0x80008000u : u; }
__device__ __forceinline__ float bf_lo(unsigned u) { return __uint_as_float(u << 16); }
__device__ __forceinline__ float bf_hi(unsigned u) { return __uint_as_float(u & 0xffff0000u); }

__device__ __forceinline__ __amdgpu_buffer_rsrc_t mk_rsrc(const void* p, unsigned bytes) {
  return __builtin_amdgcn_make_buffer_rsrc(const_cast<void*>(p), 0, bytes, 0x00020000);
}
}

__global__ void __launch_bounds__(128, 2) gemm_rs_kernel(Params p) {
  __shared__ __attribute__((aligned(16))) u32x4 lds_b[2 * 16 * 64];
  const int lane = threadIdx.x & 63;
  const int h = __builtin_amdgcn_readfirstlane(threadIdx.x >> 6);
  const int xcd = blockIdx.x & 7;
  const int idx = ((blockIdx.x >> 3) << 1) | h;
  const int wv = idx * 8 + xcd;
  const int col0 = (xcd * 4 + ((idx >> 1) & 3)) * 64;
  const int rb = ((idx >> 3) << 1) | h;
  const int rank = p.rank;
  const int l15 = lane & 15, kq = lane >> 4;

  char* myws = (char*)p.ws[rank];
  unsigned* ctr = (unsigned*)(myws + CTR_OFF + wv * 64);
  const unsigned call_v = __hip_atomic_load(ctr, __ATOMIC_RELAXED, __HIP_MEMORY_SCOPE_SYSTEM);

  const __amdgpu_buffer_rsrc_t rx = mk_rsrc(p.x, M_ * KL * 2);
  const __amdgpu_buffer_rsrc_t rw = mk_rsrc(p.w, KL * N_ * 2);
  const __amdgpu_buffer_rsrc_t rme = mk_rsrc(myws, CTR_OFF);

  const int q0 = wv % 7;
  auto dest_of = [&](int t) { int q = q0 + t; q = q >= 7 ? q - 7 : q; return t == 7 ? rank : ((rank + 1 + q) & 7); };

  const unsigned a_off = (unsigned)((rb * 16 + l15) * KL + kq * 8) * 2u;
  const unsigned a_off_lo = a_off + 256u * h, a_off_hi = a_off - 256u * h;
  u32x4 a0[8], a1[8], a2[8];
  auto load_a = [&](int t, u32x4 (&a)[8]) {
#pragma unroll
    for (int u = 0; u < 8; ++u)
      a[u] = __builtin_amdgcn_raw_buffer_load_b128(rx, (u < 4 ? a_off_lo : a_off_hi) + u * 64, dest_of(t) * (MS * KL * 2), 0);
  };
  u32x2 raw[4][8];
  {
    const unsigned b_off = (unsigned)((h * 128 + kq * 8) * N_ + col0 + 4 * l15) * 2u;
#pragma unroll
    for (int u = 0; u < 4; ++u)
#pragma unroll
      for (int i = 0; i < 8; ++i)
        raw[u][i] = __builtin_amdgcn_raw_buffer_load_b64(rw, b_off + (u * 32 + i) * (N_ * 2), 0, 0);
  }
  load_a(0, a0);
  load_a(1, a1);
  __builtin_amdgcn_sched_barrier(0);
  u32x4 bfr[8][4];
#pragma unroll
  for (int u = 0; u < 4; ++u) {
#pragma unroll
    for (int m = 0; m < 4; ++m) {
      bfr[u][0][m] = __builtin_amdgcn_perm(raw[u][2 * m + 1].x, raw[u][2 * m].x, 0x05040100u);
      bfr[u][1][m] = __builtin_amdgcn_perm(raw[u][2 * m + 1].x, raw[u][2 * m].x, 0x07060302u);
      bfr[u][2][m] = __builtin_amdgcn_perm(raw[u][2 * m + 1].y, raw[u][2 * m].y, 0x05040100u);
      bfr[u][3][m] = __builtin_amdgcn_perm(raw[u][2 * m + 1].y, raw[u][2 * m].y, 0x07060302u);
    }
#pragma unroll
    for (int j = 0; j < 4; ++j) lds_b[(h * 16 + u * 4 + j) * 64 + lane] = bfr[u][j];
  }
  asm volatile("s_waitcnt lgkmcnt(0)" ::: "memory");
  __builtin_amdgcn_s_barrier();
  asm volatile("" ::: "memory");
#pragma unroll
  for (int u = 0; u < 4; ++u)
#pragma unroll
    for (int j = 0; j < 4; ++j) bfr[4 + u][j] = lds_b[((h ^ 1) * 16 + u * 4 + j) * 64 + lane];

  const unsigned call = __builtin_amdgcn_readfirstlane(call_v) + 1u;
  const unsigned par = call & 1u;
  const unsigned my_tile = wv * TILE_BYTES + lane * 16u;

  auto tile = [&](const u32x4 (&a)[8], f32x4 (&acc)[4]) {
#pragma unroll
    for (int j = 0; j < 4; ++j) acc[j] = f32x4{0.f, 0.f, 0.f, 0.f};
#pragma unroll
    for (int s = 0; s < 8; ++s)
#pragma unroll
      for (int j = 0; j < 4; ++j)
        acc[j] = __builtin_amdgcn_mfma_f32_16x16x32_bf16(
            __builtin_bit_cast(bf16x8, a[s]), __builtin_bit_cast(bf16x8, bfr[s][j]), acc[j], 0, 0, 0);
  };
  auto push = [&](int d, const f32x4 (&acc)[4]) {
    u32x4 o0, o1;
    o0.x = enc(cvt_pk(acc[0][0], acc[1][0])); o0.y = enc(cvt_pk(acc[2][0], acc[3][0]));
    o0.z = enc(cvt_pk(acc[0][1], acc[1][1])); o0.w = enc(cvt_pk(acc[2][1], acc[3][1]));
    o1.x = enc(cvt_pk(acc[0][2], acc[1][2])); o1.y = enc(cvt_pk(acc[2][2], acc[3][2]));
    o1.z = enc(cvt_pk(acc[0][3], acc[1][3])); o1.w = enc(cvt_pk(acc[2][3], acc[3][3]));
    const __amdgpu_buffer_rsrc_t rd = mk_rsrc(p.ws[d], CTR_OFF);
    const unsigned so = par * PAR_BYTES + rank * SLOT_BYTES;
    __builtin_amdgcn_raw_buffer_store_b128(o0, rd, my_tile, so, AUX_SYS);
    __builtin_amdgcn_raw_buffer_store_b128(o1, rd, my_tile + 1024u, so, AUX_SYS);
  };

  f32x4 acc[4];
#define SB __builtin_amdgcn_sched_barrier(0)
  SB;
  tile(a0, acc); SB; load_a(2, a2); SB; push(dest_of(0), acc); SB;
  tile(a1, acc); SB; load_a(3, a0); SB; push(dest_of(1), acc); SB;
  tile(a2, acc); SB; load_a(4, a1); SB; push(dest_of(2), acc); SB;
  tile(a0, acc); SB; load_a(5, a2); SB; push(dest_of(3), acc); SB;
  tile(a1, acc); SB; load_a(6, a0); SB; push(dest_of(4), acc); SB;
  tile(a2, acc); SB; load_a(7, a1); SB; push(dest_of(5), acc); SB;
  tile(a0, acc); SB; push(dest_of(6), acc); SB;
  tile(a1, acc);
  if (lane == 0) __hip_atomic_store(ctr, call, __ATOMIC_RELAXED, __HIP_MEMORY_SCOPE_SYSTEM);

  u32x4 v[7][2];
  unsigned pending = 0x7fu;
  bool timed_out = false;
  const unsigned long long t0 = __builtin_amdgcn_s_memrealtime();
  while (true) {
#pragma unroll
    for (int t = 0; t < 7; ++t) {
      if (pending & (1u << t)) {
        int q = q0 + t; q = q >= 7 ? q - 7 : q;
        const int src = (rank + 7 - q) & 7;
        const unsigned so = par * PAR_BYTES + src * SLOT_BYTES;
        v[t][0] = __builtin_amdgcn_raw_buffer_load_b128(rme, my_tile, so, AUX_SYS);
        v[t][1] = __builtin_amdgcn_raw_buffer_load_b128(rme, my_tile + 1024u, so, AUX_SYS);
      }
    }
#pragma unroll
    for (int t = 0; t < 7; ++t) {
      if (pending & (1u << t)) {
        const bool ok = v[t][0].x && v[t][0].y && v[t][0].z && v[t][0].w &&
                        v[t][1].x && v[t][1].y && v[t][1].z && v[t][1].w;
        if (__all(ok)) pending &= ~(1u << t);
      }
    }
    if (!pending) break;
    if (__builtin_amdgcn_s_memrealtime() - t0 > DEADLINE_TICKS) { timed_out = true; break; }
    __builtin_amdgcn_s_sleep(2);
    asm volatile("" ::: "memory");
  }
  {
    const u32x4 z = {0u, 0u, 0u, 0u};
#pragma unroll
    for (int t = 0; t < 7; ++t) {
      int q = q0 + t; q = q >= 7 ? q - 7 : q;
      const int src = (rank + 7 - q) & 7;
      const unsigned so = par * PAR_BYTES + src * SLOT_BYTES;
      __builtin_amdgcn_raw_buffer_store_b128(z, rme, my_tile, so, AUX_SYS);
      __builtin_amdgcn_raw_buffer_store_b128(z, rme, my_tile + 1024u, so, AUX_SYS);
    }
  }
#pragma unroll
  for (int t = 0; t < 7; ++t) {
    acc[0][0] += bf_lo(v[t][0].x); acc[1][0] += bf_hi(v[t][0].x);
    acc[2][0] += bf_lo(v[t][0].y); acc[3][0] += bf_hi(v[t][0].y);
    acc[0][1] += bf_lo(v[t][0].z); acc[1][1] += bf_hi(v[t][0].z);
    acc[2][1] += bf_lo(v[t][0].w); acc[3][1] += bf_hi(v[t][0].w);
    acc[0][2] += bf_lo(v[t][1].x); acc[1][2] += bf_hi(v[t][1].x);
    acc[2][2] += bf_lo(v[t][1].y); acc[3][2] += bf_hi(v[t][1].y);
    acc[0][3] += bf_lo(v[t][1].z); acc[1][3] += bf_hi(v[t][1].z);
    acc[2][3] += bf_lo(v[t][1].w); acc[3][3] += bf_hi(v[t][1].w);
  }
  const __amdgpu_buffer_rsrc_t ro = mk_rsrc(p.out, MS * N_ * 2);
#pragma unroll
  for (int r = 0; r < 4; ++r) {
    u32x2 o;
    o.x = cvt_pk(acc[0][r], acc[1][r]);
    o.y = cvt_pk(acc[2][r], acc[3][r]);
    if (timed_out) { o.x = 0x7fc07fc0u; o.y = 0x7fc07fc0u; }
    __builtin_amdgcn_raw_buffer_store_b64(o, ro, (unsigned)((rb * 16 + kq * 4 + r) * N_ + col0 + 4 * l15) * 2u, 0, 0);
  }
}

extern "C" void kernel_launch(int rank, int nranks, ncclComm_t comm,
                              hipStream_t stream,
                              void* const* d_in, const int* in_sizes, int n_in,
                              void* d_out, int out_size,
                              const int* in_dtypes, int out_dtype,
                              void* const* d_out_all,
                              void* const* d_ws_all, size_t ws_bytes) {
  if (nranks != NR || ws_bytes < WS_NEED || (unsigned)rank >= (unsigned)NR) {
    fprintf(stderr, "unexpected configuration\n");
    return;
  }
  Params p;
  for (int i = 0; i < NR; ++i) p.ws[i] = d_ws_all[i];
  p.x = d_in[0];
  p.w = d_in[1];
  p.out = d_out;
  p.rank = rank;
  gemm_rs_kernel<<<NWAVES / 2, 128, 0, stream>>>(p);
  HIP_CHECK(hipGetLastError());
}
```
